```python
import numpy as np

kernel_name = "gpu_gemm_a2a_m4096_k4096_n16384_w8a8_dqsilu"


ABSMAX_RTOL = 0.001

COMPUTE_DTYPE = "int8"
EPILOGUE = "dequant_silu"
IN_QUANT = "int8"
OUT_QUANT = None
IN_DTYPES = {"x": "int8", "w_mat": "int8",
             "scale_x": "float32", "scale_w": "float32"}
WORLD_SIZE = 8

def setup_inputs():

    x = np.random.randint(-127, 128, size=(4096, 4096)).astype(np.int8)
    w_mat = np.random.randint(-127, 128, size=(4096, 16384)).astype(np.int8)
    scale_x = (0.02 * (1.0 + np.random.rand(1))).astype(np.float32)
    scale_w = (0.02 * (1.0 + np.random.rand(1))).astype(np.float32)
    return {"x": x, "w_mat": w_mat, "scale_x": scale_x, "scale_w": scale_w}

def reference(x, w_mat, scale_x, scale_w):
    acc = x.astype(np.float64) @ w_mat.astype(np.float64)
    y = (acc.astype(np.float32) * (scale_x * scale_w)).astype(np.float32)
    return (y / (1.0 + np.exp(-np.clip(y, -60.0, 60.0)))).astype(np.float32)
```

```cpp
#include <hip/hip_runtime.h>
#include <rccl/rccl.h>
#include <cstdio>
#include <type_traits>

typedef int v4i __attribute__((ext_vector_type(4)));
typedef int v2i __attribute__((ext_vector_type(2)));
typedef unsigned v4u __attribute__((ext_vector_type(4)));
typedef float v4f __attribute__((ext_vector_type(4)));
#define AS3 __attribute__((address_space(3)))

constexpr int NR = 8;
constexpr int KD = 4096, NG = 16384, NS = 2048, MS = 512;
constexpr int NBLK = 32;
constexpr size_t SLOT = (size_t)MS * KD;
constexpr size_t OFF_FLAGS = 2 * NR * SLOT;
constexpr size_t OFF_TMO = OFF_FLAGS + 16384;
constexpr size_t OFF_CTR = OFF_FLAGS + 32768;
constexpr size_t OFF_DUMMY = OFF_CTR + 1024 * 128;
constexpr size_t WS_NEED = OFF_DUMMY + 4096;
constexpr unsigned long long DEADLINE = 20000000ull;
constexpr int L_RING = 0;
constexpr int L_RED = 131072;
constexpr int L_MIR = L_RED + 24576;
constexpr int L_CNT = L_MIR + 512 + 16;
constexpr int L_TOTAL = L_MIR + 512 + 64;

struct Args {
  const char* x; const char* w; const float* sx; const float* sw; float* out;
  char* my; char* peer[7]; int rank;
};

template <int N> __device__ __forceinline__ void waitv() { asm volatile("s_waitcnt vmcnt(%0)" :: "i"(N) : "memory"); }
__device__ __forceinline__ void st16_sys(char* p, v4i v) { asm volatile("global_store_dwordx4 %0, %1, off sc0 sc1\n\ts_nop 1" :: "v"(p), "v"(v) : "memory"); }
__device__ __forceinline__ void ld16(v4i& v, const char* p) { asm volatile("global_load_dwordx4 %0, %1, off" : "=v"(v) : "v"(p) : "memory"); }
template <int IMM> __device__ __forceinline__ void glds_first(unsigned voff, const char* sbase, unsigned ldsaddr) {
  asm volatile("s_mov_b32 m0, %2\n\ts_nop 0\n\tglobal_load_lds_dwordx4 %0, %1 offset:%3" :: "v"(voff), "s"(sbase), "s"(ldsaddr), "i"(IMM) : "memory");
}
template <int IMM> __device__ __forceinline__ void glds_next(unsigned voff, const char* sbase) {
  asm volatile("global_load_lds_dwordx4 %0, %1 offset:%2" :: "v"(voff), "s"(sbase), "i"(IMM) : "memory");
}
__device__ __forceinline__ void glds16_sys(unsigned voff, const char* sbase, unsigned ldsaddr) {
  unsigned keep;
  asm volatile("s_mov_b32 %0, m0\n\ts_mov_b32 m0, %3\n\ts_nop 0\n\tglobal_load_lds_dwordx4 %1, %2 sc0 sc1\n\ts_mov_b32 m0, %0"
               : "=&s"(keep) : "v"(voff), "s"(sbase), "s"(ldsaddr) : "memory");
}
__device__ __forceinline__ void mfma_first(v4i& acc, const v4i& w, const v4i& x) {
  asm volatile("v_mfma_i32_16x16x64_i8 %0, %1, %2, 0" : "=&v"(acc) : "a"(w), "v"(x));
}
__device__ __forceinline__ void mfma_acc(v4i& acc, const v4i& w, const v4i& x) {
  asm volatile("v_mfma_i32_16x16x64_i8 %0, %1, %2, %0" : "+v"(acc) : "a"(w), "v"(x));
}
#define PIN(x) asm volatile("" : "+v"(x))
#define PINS(x) asm volatile("" : "+s"(x))
#define SCHED __builtin_amdgcn_sched_barrier(0)
__device__ __forceinline__ unsigned long long now() { return __builtin_amdgcn_s_memrealtime(); }

constexpr int NP = 2;
constexpr unsigned TPB = 16 / NP;
constexpr unsigned NTASK = NBLK * TPB;
struct PCtx { const char* src; char* dst; unsigned* fl; AS3 unsigned* cnt; AS3 unsigned* tk; unsigned call; int lane; };
__device__ __forceinline__ unsigned pdraw(const PCtx& c) {
  unsigned t = 0;
  if (c.lane == 0) t = __hip_atomic_fetch_add(c.tk, 1u, __ATOMIC_RELAXED, __HIP_MEMORY_SCOPE_WORKGROUP);
  return __builtin_amdgcn_readfirstlane(t);
}
__device__ __forceinline__ unsigned ptoff(unsigned t) { return (t / TPB) * 65536u + (t % TPB) * (NP * 1024u); }
__device__ __forceinline__ void pload(v4i (&v)[NP], const char* src, unsigned t) {
  const char* p = src + ptoff(t);
#pragma unroll
  for (int k = 0; k < NP; ++k) ld16(v[k], p + 1024 * k);
}
__device__ __forceinline__ void pstore(v4i (&v)[NP], const PCtx& c, unsigned t) {
  char* p = c.dst + ptoff(t);
#pragma unroll
  for (int k = 0; k < NP; ++k) st16_sys(p + 1024 * k, v[k]);
}
__device__ __forceinline__ bool psignal(const PCtx& c, unsigned t) {
  const unsigned j = t / TPB;
  unsigned old = 0;
  if (c.lane == 0) old = __hip_atomic_fetch_add(c.cnt + j, 1u, __ATOMIC_RELAXED, __HIP_MEMORY_SCOPE_WORKGROUP);
  old = __builtin_amdgcn_readfirstlane(old);
  if (old == TPB - 1 && c.lane == 0) __hip_atomic_store(c.fl + j * 4, c.call, __ATOMIC_RELAXED, __HIP_MEMORY_SCOPE_SYSTEM);
  return old == TPB - 1;
}
__device__ __forceinline__ void push_loop(v4i (&v)[3][NP], const PCtx& c, unsigned t0, unsigned t1) {
  unsigned tm2 = NTASK, tm1 = NTASK, ta = t0, tb = t1;
  bool sent = false;
  int tail = 0;
#define PSTEP(A, C) { \
    const unsigned tc = pdraw(c); \
    if (tc >= NTASK) { tail = A; break; } \
    pload(v[C], c.src, tc); \
    pstore(v[A], c, ta); \
    if (sent) waitv<3 * NP + 1>(); else waitv<3 * NP>(); \
    sent = false; \
    if (tm2 < NTASK) sent = psignal(c, tm2); \
    tm2 = tm1; tm1 = ta; ta = tb; tb = tc; }
  for (;;) {
    PSTEP(0, 2)
    PSTEP(1, 0)
    PSTEP(2, 1)
  }
#undef PSTEP
  if (tail == 0) pstore(v[0], c, ta); else if (tail == 1) pstore(v[1], c, ta); else pstore(v[2], c, ta);
  waitv<2 * NP>();
  if (tail == 0) pstore(v[1], c, tb); else if (tail == 1) pstore(v[2], c, tb); else pstore(v[0], c, tb);
  if (tm2 < NTASK) psignal(c, tm2);
  waitv<2 * NP>(); if (tm1 < NTASK) psignal(c, tm1);
  waitv<NP>(); psignal(c, ta);
  waitv<0>(); psignal(c, tb);
}

__global__ void __launch_bounds__(256, 1) fused_kernel(Args a) {
  extern __shared__ __attribute__((aligned(16))) char smem_[];
  AS3 char* lds = (AS3 char*)smem_;
  const int b = blockIdx.x;
  const int lane = threadIdx.x & 63;
  const int wave = __builtin_amdgcn_readfirstlane(threadIdx.x >> 6);
  const int xcd = b & 7, slot = b >> 3;
  char* my = a.my;
  unsigned callv = 0;
  {
    unsigned one = 1;
    unsigned* cp = (unsigned*)(my + OFF_CTR + (b * 4 + wave) * 128);
    if (slot >= 4 || xcd != 7)
      if (lane == 0) asm volatile("global_atomic_add %0, %1, %2, off sc0" : "=v"(callv) : "v"(cp), "v"(one) : "memory");
  }
  if (slot < 4) {
    if (xcd == 7) return;
    const int part = slot;
    const char* src = a.x + part * 16384 + 16 * lane;
    v4i v[3][NP];
    const unsigned t0 = wave, t1 = 4 + wave;
    pload(v[0], src, t0); pload(v[1], src, t1);
    AS3 unsigned* pl = (AS3 unsigned*)lds;
    if (threadIdx.x < 64) pl[threadIdx.x] = (threadIdx.x == 32) ? 8u : 0u;
    asm volatile("s_waitcnt lgkmcnt(0)" ::: "memory");
    __builtin_amdgcn_s_barrier();
    asm volatile("s_waitcnt vmcnt(0)" : "+v"(callv) :: "memory");
    const unsigned call = __builtin_amdgcn_readfirstlane(callv) + 1;
    char* peer = a.peer[0];
#pragma unroll
    for (int k = 1; k < 7; ++k) if (xcd == k) peer = a.peer[k];
    PCtx c;
    c.src = src; c.dst = peer + (size_t)((call & 1) * 8 + a.rank) * SLOT + part * 16384 + 16 * lane;
    c.fl = (unsigned*)(peer + OFF_FLAGS + a.rank * 1024 + part * 4);
    c.cnt = pl; c.tk = pl + 32; c.call = call; c.lane = lane;
    push_loop(v, c, t0, t1);
    return;
  }
  const int g = slot - 4, cg = g & 3, j = g >> 2;
  const int col0 = (xcd * 4 + cg) * 64;
  const int src = (a.rank + 1 + j) & 7;
  const int krot = cg;
  const unsigned ldsb = (unsigned)(uintptr_t)lds;
  const unsigned ring = ldsb + L_RING + wave * 32768;
  v4i wf[16][4];
  {
    const char* wb = a.w + (size_t)(wave * 1024) * NG + a.rank * NS + col0;
    const unsigned voff = (lane >> 2) * NG + (lane & 3) * 16;
    auto issue_round = [&](int r) __attribute__((always_inline)) {
      const unsigned lb = ring + (r & 1) * 16384;
#pragma unroll
      for (int p = 0; p < 16; ++p) { const int pp = (p + 2 * j) & 15; glds_first<0>(voff, wb + (size_t)(((r + krot) & 3) * 256 + pp * 16) * NG, lb + pp * 1024); }
    };
    if (threadIdx.x < 128) *(AS3 unsigned*)(lds + L_MIR + threadIdx.x * 4) = 0;
    if (threadIdx.x == 0) *(AS3 unsigned*)(lds + L_CNT) = 0;
    issue_round(0);
    issue_round(1);
    const int i = lane & 15, gq = lane >> 4;
    const int rd = (16 * gq + (i >> 1)) * 64 + 8 * (i & 1);
#pragma unroll
    for (int r = 0; r < 4; ++r) {
      if (r < 3) waitv<16>(); else waitv<0>();
      AS3 char* sb = lds + L_RING + wave * 32768 + (r & 1) * 16384;
#pragma unroll
      for (int q = 0; q < 4; ++q)
#pragma unroll
        for (int t = 0; t < 4; ++t) {
          const int ct = (t + wave) & 3;
          AS3 char* p = sb + rd + q * 4096 + ct * 16;
          v2i lo = __builtin_amdgcn_ds_read_tr8_b64_v2i32((AS3 v2i*)p);
          v2i hi = __builtin_amdgcn_ds_read_tr8_b64_v2i32((AS3 v2i*)(p + 512));
          wf[r * 4 + q][t] = v4i{lo.x, lo.y, hi.x, hi.y};
        }
      if (r + 2 < 4) {
        asm volatile("s_waitcnt lgkmcnt(0)" ::: "memory");
        asm volatile("" :: "v"(wf[r * 4 + 3][3]), "v"(wf[r * 4 + 3][0]) : "memory");
        issue_round(r + 2);
      }
    }
  }
#pragma unroll
  for (int k = 0; k < 16; ++k)
#pragma unroll
    for (int t = 0; t < 4; ++t) asm volatile("" : "+a"(wf[k][t]));
  asm volatile("s_waitcnt vmcnt(0) lgkmcnt(0)" : "+v"(callv) :: "memory");
  const unsigned call = __builtin_amdgcn_readfirstlane(callv) + 1;
  __builtin_amdgcn_s_barrier();
  const float scale = a.sx[0] * a.sw[0];
  const int nown = j < 4 ? 5 : 4, total = nown + NBLK;
  const char* const stage = my + (size_t)((call & 1) * 8 + src) * SLOT + wave * 1024;
  const int own0 = j * 4 + (j < 4 ? j : 4);
  const char* xown = a.x + wave * 1024 + (size_t)own0 * 65536;
  const int rank = a.rank;
  const char* flags = my + OFF_FLAGS + src * 1024;
  const unsigned voff_e = (lane >> 3) * KD + (((lane & 7) ^ (lane >> 4)) * 16);
  const unsigned voff_o = (8 + (lane >> 3)) * KD + (((lane & 7) ^ (4 + (lane >> 4))) * 16);
  const unsigned vo0 = voff_e + 3072, vo1 = voff_o + 2048, vo2 = voff_e + 128 + 1024, vo3 = voff_o + 128;
  const int fm = lane & 15, fg = lane >> 4;
  const int rd_e = (fm >> 3) * 1024 + (fm & 7) * 128 + ((fg ^ ((fm >> 1) & 7)) * 16);
  const int rd_o = (fm >> 3) * 1024 + (fm & 7) * 128 + (((4 + fg) ^ ((fm >> 1) & 7)) * 16);
  unsigned* tmo = (unsigned*)(my + OFF_TMO);
  AS3 char* mir = lds + L_MIR;
  const unsigned mir_addr = ldsb + L_MIR;
  int lim = nown;
  float psc = scale;
  const float nsl = -scale * 1.44269504088896f;
  const float CL = 60.0f * 1.44269504088896f;
  const v4i zero4 = {0, 0, 0, 0};
  v4i prev[4] = {zero4, zero4, zero4, zero4};
  v4i xn[4] = {zero4, zero4, zero4, zero4};
  const char* prev_ob = my + OFF_DUMMY;
  unsigned out_voff = threadIdx.x * 16;
  const unsigned out_voff_real = ((lane & 15) * NS + col0 + wave * 16 + (lane >> 4) * 4) * 4;
  AS3 char* const fe = lds + L_RING + wave * 32768 + rd_e;
  AS3 char* const fo = lds + L_RING + wave * 32768 + rd_o;
  AS3 char* const redr = lds + L_RED + wave * 3072 + lane * 16;
  AS3 char* const redw1 = lds + L_RED + ((1 + wave) & 3) * 3072 + lane * 16;
  AS3 char* const redw2 = lds + L_RED + ((2 + wave) & 3) * 3072 + 1024 + lane * 16;
  AS3 char* const redw3 = lds + L_RED + ((3 + wave) & 3) * 3072 + 2048 + lane * 16;
  AS3 char* const mirl = mir + (lane & 31) * 16;
  const unsigned cnt_addr = ldsb + L_CNT;
  constexpr unsigned POISON = 1u << 30;

  auto lim_from = [&](v4u f) __attribute__((always_inline)) {
    const unsigned mn = min(min(f.x, f.y), min(f.z, f.w));
    const unsigned m = (unsigned)__ballot(mn >= call);
    return nown + ((m == 0xffffffffu) ? 32 : __builtin_ctz(~m));
  };
  auto poll = [&]() __attribute__((always_inline)) { if (lane < 32) glds16_sys(lane * 16, flags, mir_addr); };
  auto blk_base = [&](int i) __attribute__((always_inline)) -> const char* {
    return (i < nown ? xown : stage) + (unsigned)(i < nown ? i : i - nown) * 65536u;
  };
  auto blk_out = [&](int i) __attribute__((always_inline)) -> const char* {
    const unsigned row0 = i < nown ? (unsigned)(rank * MS + 16 * (own0 + i)) : (unsigned)(src * MS + 16 * (i - nown));
    return (const char*)a.out + (size_t)row0 * (NS * 4);
  };
  const char* nx_gb = nullptr; const char* nx_ob = nullptr;
  const int kq0 = ((0 + krot) & 3) * 256 - 3072, kq1 = ((1 + krot) & 3) * 256 - 3072, kq2 = ((2 + krot) & 3) * 256 - 3072, kq3 = ((3 + krot) & 3) * 256 - 3072;
#define KQ(q) ((q) == 0 ? kq0 : (q) == 1 ? kq1 : (q) == 2 ? kq2 : kq3)
#define PIECE(gb, lb, k) do { \
    if (((k) & 3) == 0) glds_first<0>(vo0, (gb) + KQ((k) >> 2), (lb) + ((k) >> 2) * 4096); \
    else if (((k) & 3) == 1) glds_next<1024>(vo1, (gb) + KQ((k) >> 2)); \
    else if (((k) & 3) == 2) glds_next<2048>(vo2, (gb) + KQ((k) >> 2)); \
    else glds_next<3072>(vo3, (gb) + KQ((k) >> 2)); } while (0)
  auto issue = [&](int i) __attribute__((always_inline)) {
    const char* gb = blk_base(i);
    const unsigned lb = ring + (i & 1) * 16384;
    PIECE(gb, lb, 0); PIECE(gb, lb, 1); PIECE(gb, lb, 2); PIECE(gb, lb, 3); PIECE(gb, lb, 4); PIECE(gb, lb, 5); PIECE(gb, lb, 6); PIECE(gb, lb, 7);
    PIECE(gb, lb, 8); PIECE(gb, lb, 9); PIECE(gb, lb, 10); PIECE(gb, lb, 11); PIECE(gb, lb, 12); PIECE(gb, lb, 13); PIECE(gb, lb, 14); PIECE(gb, lb, 15);
  };
  auto cnt_add = [&](unsigned v) __attribute__((always_inline)) {
    asm volatile("s_mov_b64 exec, 1\n\tds_add_u32 %0, %1\n\ts_mov_b64 exec, -1" :: "v"(cnt_addr), "v"(v) : "memory");
  };
  auto wait_cnt = [&](unsigned need, unsigned seen) __attribute__((always_inline)) {
    unsigned c = __builtin_amdgcn_readfirstlane(seen);
    if (__builtin_expect(c < need, 0)) {
      do { __builtin_amdgcn_s_sleep(1); c = __builtin_amdgcn_readfirstlane(*(volatile AS3 unsigned*)(lds + L_CNT)); } while (c < need);
    }
    if (__builtin_expect(c >= POISON, 0)) psc = __builtin_nanf("");
  };
  auto slow_wait = [&](int i) __attribute__((always_inline)) {
    const unsigned long long t0 = now();
    unsigned it = 0;
    while (true) {
      if (wave == 0) { poll(); waitv<0>(); } else __builtin_amdgcn_s_sleep(1);
      lim = lim_from(*(volatile AS3 v4u*)mirl);
      if (i < lim) break;
      if (__builtin_amdgcn_readfirstlane(*(volatile AS3 unsigned*)(lds + L_CNT)) >= POISON) { lim = total; break; }
      if ((++it & 31) == 0) {
        if (__hip_atomic_load(tmo, __ATOMIC_RELAXED, __HIP_MEMORY_SCOPE_AGENT) == call || now() - t0 > DEADLINE) {
          __hip_atomic_store(tmo, call, __ATOMIC_RELAXED, __HIP_MEMORY_SCOPE_AGENT);
          cnt_add(POISON); lim = total; break;
        }
      }
    }
  };

  auto body = [&](int i, auto PAR, auto MODE_, const char* gb, const char* ob, bool tailnop) __attribute__((always_inline)) {
    constexpr int par = decltype(PAR)::value, MODE = decltype(MODE_)::value;
    constexpr bool pre = MODE == 0, chain = MODE == 0, pre1 = MODE == 2, nxt = MODE <= 1;
    constexpr int so = par * 16384, sn = 16384 - so;
    constexpr int ro = 12288 - par * 12288;
    const unsigned lb = ring + so, lbn = ring + sn;
    v4i xf[16];
    v4i acc[4];
    v4i part[3];
    v4i sum; v4f y, e; v4u mf;
    unsigned seen = 0;
    asm volatile("; body begin %0" :: "i"(MODE * 2 + par) : "memory");
    xf[0] = xn[0]; xf[1] = xn[1]; xf[2] = xn[2]; xf[3] = xn[3];
    SCHED;
#define MF(k, t) do { if ((k) == 0) mfma_first(acc[t], wf[k][t], xf[k]); else mfma_acc(acc[t], wf[k][t], xf[k]); } while (0)
#define NSAFE(k) ((10 - (k)) + (nxt ? 16 : 0) + (pre && (k) > 0 ? (k) - 1 : 0) + (pre1 ? (2 * (k) < 16 ? 2 * (k) : 16) : 0))
#define RD(k) { if ((k) < 12 && ((k) & 1) == 0) waitv<NSAFE(k)>(); \
      if ((k) + 4 < 16) xf[((k) + 4) & 15] = *(AS3 v4i*)((((k) & 1) ? fo : fe) + so + (((k) + 4) >> 1) * 2048); }
#define GL(k) { if (pre && (k) >= 1) { \
        asm volatile("" :: "v"(xf[((k) - 1) & 14]), "v"(xf[(((k) - 1) & 14) + 1]) : "memory"); \
        PIECE(gb, lb, ((k) - 1) & 15); } \
      if (pre1 && (k) < 8) PIECE(gb, lbn, (2 * (k)) & 15); }
#define GL2(k) { if (pre1 && (k) < 8) PIECE(gb, lbn, (2 * (k) + 1) & 15); }
#define STEP(k, X0, X1, X2, X3) \
    { MF(k, 0); RD(k); X0; SCHED; MF(k, 1); GL(k); X1; SCHED; MF(k, 2); X2; SCHED; MF(k, 3); GL2(k); X3; SCHED; }
#define SIG(c) e.c = __builtin_amdgcn_fmed3f(e.c, -CL, CL)
    STEP(0, , , { PIN(prev[0]); PIN(prev[1]); PIN(prev[2]); *(AS3 v4i*)(redw1 + ro) = prev[1]; }, { *(AS3 v4i*)(redw2 + ro) = prev[2]; })
    STEP(1, , , { PIN(prev[3]); *(AS3 v4i*)(redw3 + ro) = prev[3]; }, { cnt_add(1); })
    STEP(2, , , { if (pre) { nx_gb = blk_base(i + 3); PINS(nx_gb); } }, )
    STEP(3, , , { if (pre) { nx_ob = blk_out(i + 1); PINS(nx_ob); } }, )
    STEP(4, , , , { if (wave == 0 && lim < total) poll(); })
    STEP(5, , , { seen = *(volatile AS3 unsigned*)(lds + L_CNT); }, )
    STEP(6, , , , { wait_cnt(4u * (unsigned)(i + 1), seen);
                    part[0] = *(AS3 v4i*)(redr + ro); part[1] = *(AS3 v4i*)(redr + ro + 1024); part[2] = *(AS3 v4i*)(redr + ro + 2048); })
    STEP(7, , , , )
    STEP(8, , , { sum.x = prev[0].x + part[0].x + part[1].x; sum.y = prev[0].y + part[0].y + part[1].y; PIN(sum); },
                { sum.z = prev[0].z + part[0].z + part[1].z; sum.w = prev[0].w + part[0].w + part[1].w; PIN(sum); })
    STEP(9, , , { sum.x += part[2].x; sum.y += part[2].y; PIN(sum); }, { sum.z += part[2].z; sum.w += part[2].w; PIN(sum); })
    STEP(10, { y.x = (float)sum.x; PIN(y.x); }, { y.y = (float)sum.y; PIN(y.y); }, { y.z = (float)sum.z; y.w = (float)sum.w; PIN(y.z); PIN(y.w); },
             { e.x = y.x * nsl; e.y = y.y * nsl; PIN(e.x); PIN(e.y); })
    STEP(11, { e.z = y.z * nsl; PIN(e.z); }, { e.w = y.w * nsl; PIN(e.w); }, { SIG(x); SIG(y); PIN(e.x); PIN(e.y); }, { SIG(z); SIG(w); PIN(e.z); PIN(e.w); })
    STEP(12, { if (chain) { waitv<12 + (pre ? 11 : 0)>(); xn[0] = *(AS3 v4i*)(fe + sn); } }, { e.x = __builtin_amdgcn_exp2f(e.x); PIN(e.x); },
             { if (chain) xn[1] = *(AS3 v4i*)(fo + sn); e.y = __builtin_amdgcn_exp2f(e.y); PIN(e.y); }, { e.z = __builtin_amdgcn_exp2f(e.z); y.x *= psc; PIN(e.z); PIN(y.x); })
    STEP(13, { if (chain) xn[2] = *(AS3 v4i*)(fe + sn + 2048); }, { e.w = __builtin_amdgcn_exp2f(e.w); PIN(e.w); },
             { if (chain) xn[3] = *(AS3 v4i*)(fo + sn + 2048); e.x += 1.0f; e.y += 1.0f; PIN(e.x); PIN(e.y); }, { mf = *(volatile AS3 v4u*)mirl; e.z += 1.0f; e.w += 1.0f; PIN(e.z); PIN(e.w); })
    STEP(14, { e.x = __builtin_amdgcn_rcpf(e.x); PIN(e.x); }, { e.y = __builtin_amdgcn_rcpf(e.y); PIN(e.y); },
             { e.z = __builtin_amdgcn_rcpf(e.z); y.y *= psc; PIN(e.z); PIN(y.y); }, { e.w = __builtin_amdgcn_rcpf(e.w); y.z *= psc; PIN(e.w); PIN(y.z); })
    STEP(15, { y.w *= psc; y.x *= e.x; PIN(y.w); PIN(y.x); }, { y.y *= e.y; PIN(y.y); }, { y.z *= e.z; y.w *= e.w; PIN(y.z); PIN(y.w); },
             { asm volatile("global_store_dwordx4 %0, %1, %2\n\ts_nop 1" :: "v"(out_voff), "v"(y), "s"(prev_ob) : "memory");
               lim = max(lim, lim_from(mf)); })
#undef SIG
#undef GL2
#undef GL
#undef RD
#undef STEP
#undef NSAFE
#undef MF
    if (pre) PIECE(gb, lb, 15);
    if (tailnop) asm volatile("s_nop 7\n\ts_nop 7" : "+v"(acc[0]), "+v"(acc[1]), "+v"(acc[2]), "+v"(acc[3]));
    asm volatile("; body end %0" :: "i"(MODE * 2 + par) : "memory");
    prev[0] = acc[0]; prev[1] = acc[1]; prev[2] = acc[2]; prev[3] = acc[3];
    prev_ob = ob; out_voff = out_voff_real;
  };
  issue(0); issue(1);
  waitv<28>();
  xn[0] = *(AS3 v4i*)(fe); xn[1] = *(AS3 v4i*)(fo); xn[2] = *(AS3 v4i*)(fe + 2048); xn[3] = *(AS3 v4i*)(fo + 2048);
  int i = 0;
  auto load_xn = [&](auto PAR) __attribute__((always_inline)) {
    constexpr int so = decltype(PAR)::value * 16384;
    xn[0] = *(AS3 v4i*)(fe + so); xn[1] = *(AS3 v4i*)(fo + so); xn[2] = *(AS3 v4i*)(fe + so + 2048); xn[3] = *(AS3 v4i*)(fo + so + 2048);
  };
  using P0 = std::integral_constant<int, 0>; using P1 = std::integral_constant<int, 1>;
  using M_PRE = std::integral_constant<int, 0>; using M_DRAIN = std::integral_constant<int, 1>;
  using M_PRE1 = std::integral_constant<int, 2>; using M_LAST = std::integral_constant<int, 3>;
#define TAILNOP asm volatile("s_nop 7\n\ts_nop 7" : "+v"(prev[0]), "+v"(prev[1]), "+v"(prev[2]), "+v"(prev[3]))
  for (;;) {
    if (i + 3 < lim) {
      const char* gb = blk_base(i + 2); const char* ob = blk_out(i);
      do {
        body(i, P0{}, M_PRE{}, gb, ob, false);
        body(i + 1, P1{}, M_PRE{}, nx_gb, nx_ob, false);
        gb = nx_gb; ob = nx_ob;
        i += 2;
      } while (i + 3 < lim);
    }
    TAILNOP;
    body(i, P0{}, M_DRAIN{}, nullptr, blk_out(i), true);
    ++i;
    bool done = false;
    for (;;) {
      if (i + 1 == total) { waitv<12>(); load_xn(P1{}); body(i, P1{}, M_LAST{}, nullptr, blk_out(i), true); done = true; break; }
      if (i + 1 >= lim) slow_wait(i + 1);
      if (i + 4 < lim) break;
      waitv<12>(); load_xn(P1{});
      body(i, P1{}, M_PRE1{}, blk_base(i + 1), blk_out(i), false);
      ++i;
      if (i + 1 == total) { TAILNOP; waitv<12>(); load_xn(P0{}); body(i, P0{}, M_LAST{}, nullptr, blk_out(i), true); done = true; break; }
      if (i + 1 >= lim) slow_wait(i + 1);
      waitv<12>(); load_xn(P0{});
      body(i, P0{}, M_PRE1{}, blk_base(i + 1), blk_out(i), false);
      ++i;
    }
    if (done) break;
    TAILNOP;
    issue(i + 1);
    waitv<28>(); load_xn(P1{});
    body(i, P1{}, M_PRE{}, blk_base(i + 2), blk_out(i), true);
    ++i;
  }
#undef TAILNOP
  {
    const int ro = ((total - 1) & 1) * 12288;
    *(AS3 v4i*)(redw1 + ro) = prev[1]; *(AS3 v4i*)(redw2 + ro) = prev[2]; *(AS3 v4i*)(redw3 + ro) = prev[3];
    cnt_add(1);
    wait_cnt(4u * (unsigned)(total + 1), 0);
    v4i s = prev[0] + *(AS3 v4i*)(redr + ro) + *(AS3 v4i*)(redr + ro + 1024) + *(AS3 v4i*)(redr + ro + 2048);
    v4f o;
    const int sv[4] = {s.x, s.y, s.z, s.w};
#pragma unroll
    for (int q = 0; q < 4; ++q) {
      const float t = (float)sv[q];
      const float e = __builtin_amdgcn_exp2f(__builtin_amdgcn_fmed3f(t * nsl, -CL, CL));
      o[q] = t * psc * __builtin_amdgcn_rcpf(1.0f + e);
    }
    asm volatile("global_store_dwordx4 %0, %1, %2\n\ts_nop 1" :: "v"(out_voff), "v"(o), "s"(prev_ob) : "memory");
  }
}

extern "C" void kernel_launch(int rank, int nranks, ncclComm_t comm, hipStream_t stream,
                              void* const* d_in, const int* in_sizes, int n_in,
                              void* d_out, int out_size, const int* in_dtypes, int out_dtype,
                              void* const* d_out_all, void* const* d_ws_all, size_t ws_bytes) {
  if (nranks != NR || ws_bytes < WS_NEED || n_in < 4) { fprintf(stderr, "unexpected configuration\n"); return; }
  static bool init = false;
  if (!init) {
    hipError_t e = hipFuncSetAttribute((const void*)fused_kernel, hipFuncAttributeMaxDynamicSharedMemorySize, L_TOTAL);
    if (e != hipSuccess) fprintf(stderr, "hipFuncSetAttribute: %s\n", hipGetErrorString(e));
    init = true;
  }
  Args a;
  a.x = (const char*)d_in[0]; a.w = (const char*)d_in[1]; a.sx = (const float*)d_in[2]; a.sw = (const float*)d_in[3];
  a.out = (float*)d_out; a.my = (char*)d_ws_all[rank]; a.rank = rank;
  for (int d = 0; d < 7; ++d) a.peer[d] = (char*)d_ws_all[(rank + 1 + d) & 7];
  fused_kernel<<<256, 256, L_TOTAL, stream>>>(a);
}
```
